```python
import numpy as np

kernel_name = "gpu_gemm_a2a_m4096_k4096_n4096_e4m3e4m3_dequant"


ABSMAX_RTOL = 0.03

COMPUTE_DTYPE = "float32"
EPILOGUE = "dequant"
IN_QUANT = None
OUT_QUANT = None
IN_FP8 = "e4m3e4m3"
IN_DTYPES = {"x": "e4m3", "w_mat": "e4m3",
             "scale_x": "float32", "scale_w": "float32"}
WORLD_SIZE = 8

def _snap_fp8(x, mant_bits, max_abs, min_normal):
    a = np.abs(x.astype(np.float64))
    _m, _e = np.frexp(a)
    q = np.where(a >= min_normal, np.ldexp(1.0, _e - 1 - mant_bits),
                 min_normal * 2.0 ** -mant_bits)
    snapped = np.minimum(np.round(a / q) * q, max_abs)
    return (np.sign(x) * snapped).astype(np.float32)

def setup_inputs():

    x = _snap_fp8(np.random.randn(4096, 4096).astype(np.float32), 3, 448.0, 2.0 ** -6)
    w_mat = _snap_fp8(np.random.randn(4096, 4096).astype(np.float32), 3, 448.0, 2.0 ** -6)
    scale_x = (0.02 * (1.0 + np.random.rand(1))).astype(np.float32)
    scale_w = (0.02 * (1.0 + np.random.rand(1))).astype(np.float32)
    return {"x": x, "w_mat": w_mat, "scale_x": scale_x, "scale_w": scale_w}

def reference(x, w_mat, scale_x, scale_w):
    acc = (x @ w_mat).astype(np.float32)
    y = (acc * (scale_x * scale_w)).astype(np.float32)
    return y
```

```cpp
#include <hip/hip_runtime.h>
#include <rccl/rccl.h>
#include <cstdio>
#include <cstdint>

#define HIP_CHECK(expr) do { hipError_t _e = (expr); if (_e != hipSuccess) { \
    fprintf(stderr, "HIP error %s at %s:%d\n", hipGetErrorString(_e), __FILE__, __LINE__); \
    return; } } while (0)

#define NX_ROWS 64

namespace {

constexpr int NR = 8;
constexpr int MS = 512;
constexpr int KD = 4096;
constexpr int ND = 4096;
constexpr int NBLK = 512;
constexpr int NX = NX_ROWS;
constexpr int BK = 128;
constexpr int NST = KD / BK;
constexpr int D = 6;
constexpr int W_BYTES = BK * 128;
constexpr int A_BYTES = 64 * BK;
constexpr int SLOT = W_BYTES + A_BYTES;
constexpr int CTRL_OFF = D * SLOT;
constexpr int LDS_BYTES = CTRL_OFF + 64;
constexpr int NWG = 256;
constexpr int NTHREADS = 9 * 64;

constexpr size_t WS_CNT_MAIN = 0;
constexpr size_t WS_CNT_COMM = 1024;
constexpr size_t WS_TMO = 2048;
constexpr size_t WS_OFLAG = 4096;
constexpr size_t WS_XFLAG = 8192;
constexpr size_t WS_XSTAGE = 65536;
constexpr size_t XSTAGE_SRC = (size_t)(NX > 0 ? NX : 1) * KD;
constexpr size_t WS_NEED = WS_XSTAGE + 2 * NR * XSTAGE_SRC;
constexpr unsigned long long DEADLINE = 50000000ull;

static_assert(NX % 32 == 0 && NX <= 64, "NX");
constexpr int RUNITS = (MS - NX) / 32;
constexpr int N1 = 16 - RUNITS;

typedef int v2i __attribute__((ext_vector_type(2)));
typedef int v4i __attribute__((ext_vector_type(4)));
typedef int v8i __attribute__((ext_vector_type(8)));
typedef unsigned v4u __attribute__((ext_vector_type(4)));
typedef float v16f __attribute__((ext_vector_type(16)));
typedef __attribute__((address_space(3))) char* ldsp;

struct Params {
    const char* x;
    const char* w;
    const float* sx;
    const float* sw;
    char* out[NR];
    char* ws[NR];
    int rank;
};

__device__ __forceinline__ unsigned long long rtc() { return __builtin_amdgcn_s_memrealtime(); }
#define STAMP(idx) do {} while (0)

#define WAIT_VM(n) asm volatile("s_waitcnt vmcnt(%0)" :: "i"(n) : "memory")
#define WAIT_LGKM0() asm volatile("s_waitcnt lgkmcnt(0)" ::: "memory")
#define BAR() __builtin_amdgcn_s_barrier()

__device__ __forceinline__ __amdgpu_buffer_rsrc_t mkrsrc(const void* base, unsigned bytes) {
    return __builtin_amdgcn_make_buffer_rsrc(const_cast<void*>(base), 0, (int)bytes, 0x00020000);
}

__device__ __forceinline__ bool wait_flags(const Params& p, size_t region, int src, bool all_src, unsigned call,
                                           int lane) {
    char* ws = p.ws[p.rank];
    unsigned* tmo = (unsigned*)(ws + WS_TMO);
    __amdgpu_buffer_rsrc_t r = mkrsrc(ws + region, 1024);
    bool mine = all_src ? ((lane >> 3) != p.rank) : ((lane >> 3) == src);
    unsigned long long t0 = rtc();
    for (;;) {
        v4u f = __builtin_amdgcn_raw_buffer_load_b128(r, lane * 16, 0, 17);
        bool ok = !mine || (f.x >= call && f.y >= call && f.z >= call && f.w >= call);
        if (__all(ok)) return true;
        if (__hip_atomic_load(tmo, __ATOMIC_RELAXED, __HIP_MEMORY_SCOPE_AGENT) == call || rtc() - t0 > DEADLINE) {
            if (lane == 0) __hip_atomic_store(tmo, call, __ATOMIC_RELAXED, __HIP_MEMORY_SCOPE_AGENT);
            return false;
        }
        __builtin_amdgcn_s_sleep(1);
    }
}

template <int MT>
__device__ __forceinline__ void load_stage(__amdgpu_buffer_rsrc_t rw, __amdgpu_buffer_rsrc_t ra, ldsp lds, int slot,
                                           int ks, int lw, unsigned vW, unsigned vA) {
    ldsp sw = lds + slot * SLOT + lw * 4096;
    unsigned so = (unsigned)ks * (BK * ND) + lw * (32 * ND);
#pragma unroll
    for (int j = 0; j < 4; ++j)
        __builtin_amdgcn_raw_ptr_buffer_load_lds(rw, (__attribute__((address_space(3))) void*)(sw + j * 1024), 16, vW,
                                                 so + j * (8 * ND), 0, 0);
    ldsp sa = lds + slot * SLOT + W_BYTES + lw * 1024;
    unsigned sao = (unsigned)ks * BK + lw * (8 * KD);
#pragma unroll
    for (int m = 0; m < MT; ++m)
        __builtin_amdgcn_raw_ptr_buffer_load_lds(ra, (__attribute__((address_space(3))) void*)(sa + m * 4096), 16, vA,
                                                 sao + m * (32 * KD), 0, 0);
}

template <int MT>
__device__ __forceinline__ void loader_tile(const char* wbase, const char* abase, ldsp lds, int k0, int lw, int lane) {
    constexpr int NL = 4 + MT;
    __amdgpu_buffer_rsrc_t rw = mkrsrc(wbase, (unsigned)((KD - 1) * ND + 128));
    __amdgpu_buffer_rsrc_t ra = mkrsrc(abase, (unsigned)(MT * 32 * KD));
    unsigned vW = (lane >> 3) * ND + (((lane & 7) ^ (((lane >> 4) & 3) << 1)) << 4);
    unsigned vA = (lane >> 3) * KD + (((lane & 7) ^ (((lw & 1) << 2) | (lane >> 4))) << 4);
#pragma unroll
    for (int s = 0; s < D - 1; ++s) load_stage<MT>(rw, ra, lds, s, (k0 + s) & (NST - 1), lw, vW, vA);
    int slot = D - 1;
#pragma unroll 1
    for (int s = 0; s < NST - (D - 1); ++s) {
        WAIT_VM((D - 2) * NL);
        BAR();
        load_stage<MT>(rw, ra, lds, slot, (k0 + s + D - 1) & (NST - 1), lw, vW, vA);
        slot = slot == D - 1 ? 0 : slot + 1;
    }
    WAIT_VM(4 * NL); BAR();
    WAIT_VM(3 * NL); BAR();
    WAIT_VM(2 * NL); BAR();
    WAIT_VM(1 * NL); BAR();
    WAIT_VM(0); BAR();
    static_assert(D == 6, "tail written for D == 6");
}

template <int MT>
struct Frags {
    v8i a[MT][2];
    v8i b[2];
};

template <int MT>
__device__ __forceinline__ void read_frags(Frags<MT>& f, ldsp sbase, unsigned bOff, unsigned aOff) {
    typedef __attribute__((address_space(3))) v2i* p2;
    typedef __attribute__((address_space(3))) v4i* p4;
#pragma unroll
    for (int s = 0; s < 2; ++s) {
        v2i t0 = __builtin_amdgcn_ds_read_tr8_b64_v2i32((p2)(sbase + bOff + (64 * s + 0) * 128));
        v2i t1 = __builtin_amdgcn_ds_read_tr8_b64_v2i32((p2)(sbase + bOff + (64 * s + 8) * 128));
        v2i t2 = __builtin_amdgcn_ds_read_tr8_b64_v2i32((p2)(sbase + bOff + (64 * s + 16) * 128));
        v2i t3 = __builtin_amdgcn_ds_read_tr8_b64_v2i32((p2)(sbase + bOff + (64 * s + 24) * 128));
        f.b[s] = v8i{t0.x, t0.y, t1.x, t1.y, t2.x, t2.y, t3.x, t3.y};
#pragma unroll
        for (int m = 0; m < MT; ++m) {
            v4i lo = *(p4)(sbase + W_BYTES + m * 4096 + (aOff ^ (unsigned)(64 * s)));
            v4i hi = *(p4)(sbase + W_BYTES + m * 4096 + (aOff ^ (unsigned)(64 * s + 16)));
            f.a[m][s] = v8i{lo.x, lo.y, lo.z, lo.w, hi.x, hi.y, hi.z, hi.w};
        }
    }
}

template <int MT>
__device__ __forceinline__ void mma(v16f (&acc)[MT], const Frags<MT>& f) {
#pragma unroll
    for (int s = 0; s < 2; ++s)
#pragma unroll
        for (int m = 0; m < MT; ++m)
            acc[m] = __builtin_amdgcn_mfma_scale_f32_32x32x64_f8f6f4(f.a[m][s], f.b[s], acc[m], 0, 0, 0, 0, 0, 0);
}

template <int MT>
__device__ __forceinline__ void consumer_tile(const Params& p, ldsp lds, v16f (&acc)[MT], int cw, int lane,
                                              bool publish_call, unsigned callv) {
    int g = lane >> 4, q = (lane & 15) >> 1, pp = lane & 1;
    unsigned bOff = (32 * (g >> 1) + q) * 128 + (((2 * cw + (g & 1)) ^ ((q >> 1) << 1)) << 4) + 8 * pp;
    int r = lane & 31, h = lane >> 5;
    unsigned aOff = r * 128 + (((2 * h) ^ ((r >> 1) & 7)) << 4);
#pragma unroll
    for (int m = 0; m < MT; ++m)
#pragma unroll
        for (int i = 0; i < 16; ++i) acc[m][i] = 0.f;
    Frags<MT> f0, f1;
    BAR();
    read_frags<MT>(f0, lds, bOff, aOff);
    WAIT_LGKM0();
    __builtin_amdgcn_sched_barrier(0);
    if (publish_call) {
        if (lane == 0) {
            *(__attribute__((address_space(3))) unsigned*)(lds + CTRL_OFF) = callv + 1;
            __hip_atomic_store((unsigned*)(p.ws[p.rank] + WS_CNT_MAIN) + blockIdx.x, callv + 1, __ATOMIC_RELAXED,
                               __HIP_MEMORY_SCOPE_SYSTEM);
        }
        WAIT_LGKM0();
    }
    int slot = 1;
#pragma unroll 1
    for (int s = 1; s < NST - 1; s += 2) {
        BAR();
        read_frags<MT>(f1, lds + slot * SLOT, bOff, aOff);
        mma<MT>(acc, f0);
        WAIT_LGKM0();
        __builtin_amdgcn_sched_barrier(0);
        slot = slot == D - 1 ? 0 : slot + 1;
        BAR();
        read_frags<MT>(f0, lds + slot * SLOT, bOff, aOff);
        mma<MT>(acc, f1);
        WAIT_LGKM0();
        __builtin_amdgcn_sched_barrier(0);
        slot = slot == D - 1 ? 0 : slot + 1;
    }
    BAR();
    read_frags<MT>(f1, lds + slot * SLOT, bOff, aOff);
    mma<MT>(acc, f0);
    WAIT_LGKM0();
    __builtin_amdgcn_sched_barrier(0);
    mma<MT>(acc, f1);
}

template <int MT>
__device__ __forceinline__ void store_tile(char* obase, const v16f (&acc)[MT], float scale, int cw, int lane,
                                           bool poison) {
    __amdgpu_buffer_rsrc_t ro = mkrsrc(obase, (unsigned)((MT * 32 - 1) * (NBLK * 4) + 512));
    unsigned vo = (lane >> 5) * (4 * NBLK * 4) + (32 * cw + (lane & 31)) * 4;
#pragma unroll
    for (int m = 0; m < MT; ++m)
#pragma unroll
        for (int i = 0; i < 16; ++i) {
            float v = poison ? __builtin_nanf("") : acc[m][i] * scale;
            __builtin_amdgcn_raw_buffer_store_b32(__builtin_bit_cast(unsigned, v), ro, vo,
                                                  (32 * m + (i & 3) + 8 * (i >> 2)) * (NBLK * 4), 17);
        }
}

template <int MT>
__device__ __forceinline__ void run_consumer(const Params& p, ldsp lds, char* obase, int cw, int lane, bool publish,
                                             unsigned callv, bool end_bar, bool poison) {
    v16f acc[MT];
    consumer_tile<MT>(p, lds, acc, cw, lane, publish, callv);
    if (end_bar) BAR();
    float scale = *p.sx * *p.sw;
    store_tile<MT>(obase, acc, scale, cw, lane, poison);
}

__global__ void __launch_bounds__(NTHREADS) fused_gemm_a2a(Params p) {
    __shared__ __attribute__((aligned(16))) char smem[LDS_BYTES];
    ldsp lds = (ldsp)smem;
    const int lane = threadIdx.x & 63;
    const int wave = __builtin_amdgcn_readfirstlane(threadIdx.x >> 6);
    const int b = blockIdx.x;
    const int sl = b & 7, li = b >> 3, cg = li & 3, rg = li >> 2;
    const int rank = p.rank;
    const int dest = (rank + 1 + sl) & 7;
    const bool local = sl == 7;
    const int k0 = (sl * 4) & (NST - 1);
    char* myws = p.ws[rank];

    int mt, row0;
    if (local) { mt = 2; row0 = rg * 64; }
    else if (rg < N1) { mt = 1; row0 = rg * 32; }
    else { mt = 2; row0 = N1 * 32 + (rg - N1) * 64; }
    const char* wbase = p.w + dest * NBLK + cg * 128;
    const bool second = NX > 0 && local && rg < NR - 1;
    const int src2 = (rank + 1 + rg) & 7;

    if (wave >= 4 && wave < 8) {
        const int lw = wave - 4;
        const char* abase = p.x + (size_t)row0 * KD;
        if (mt == 1) loader_tile<1>(wbase, abase, lds, k0, lw, lane);
        else loader_tile<2>(wbase, abase, lds, k0, lw, lane);
        if (NX > 0 && second) {
            unsigned call = *(__attribute__((address_space(3))) unsigned*)(lds + CTRL_OFF);
            call = __builtin_amdgcn_readfirstlane(call);
            if (lw == 0) {
                bool ok = wait_flags(p, WS_XFLAG, src2, false, call, lane);
                __builtin_amdgcn_fence(__ATOMIC_ACQUIRE, "");
                if (lane == 0) *(__attribute__((address_space(3))) unsigned*)(lds + CTRL_OFF + 4) = ok ? 0u : 1u;
                asm volatile("s_waitcnt vmcnt(0) lgkmcnt(0)" ::: "memory");
            }
            BAR();
            const char* a2 = myws + WS_XSTAGE + ((size_t)(call & 1) * NR + src2) * XSTAGE_SRC;
            loader_tile<(NX > 0 ? NX / 32 : 1)>(wbase, a2, lds, k0, lw, lane);
        }
        return;
    }
    if (wave == 8) {
        if (NX > 0 && !local) {
            unsigned* cnt = (unsigned*)(myws + WS_CNT_COMM) + b;
            unsigned call = __hip_atomic_load(cnt, __ATOMIC_RELAXED, __HIP_MEMORY_SCOPE_SYSTEM) + 1;
            constexpr int NI = NX > 0 ? NX / 8 : 1;
            __amdgpu_buffer_rsrc_t rs = mkrsrc(p.x + (size_t)(MS - NX) * KD + (size_t)li * (NI * 1024), NI * 1024);
            v4u v[NI];
#pragma unroll
            for (int j = 0; j < NI; ++j) v[j] = __builtin_amdgcn_raw_buffer_load_b128(rs, lane * 16 + j * 1024, 0, 0);
            call = __builtin_amdgcn_readfirstlane(call);
            char* dws = p.ws[dest];
            __amdgpu_buffer_rsrc_t rd =
                mkrsrc(dws + WS_XSTAGE + ((size_t)(call & 1) * NR + rank) * XSTAGE_SRC + (size_t)li * (NI * 1024),
                       NI * 1024);
#pragma unroll
            for (int j = 0; j < NI; ++j) __builtin_amdgcn_raw_buffer_store_b128(v[j], rd, lane * 16 + j * 1024, 0, 17);
            STAMP(8);
            WAIT_VM(0);
            if (lane == 0) {
                __hip_atomic_store((unsigned*)(dws + WS_XFLAG) + rank * 32 + li, call, __ATOMIC_RELAXED,
                                   __HIP_MEMORY_SCOPE_SYSTEM);
                __hip_atomic_store(cnt, call, __ATOMIC_RELAXED, __HIP_MEMORY_SCOPE_SYSTEM);
            }
            STAMP(9);
        }
        return;
    }

    const int cw = wave;
    if (cw == 0) STAMP(0);
    unsigned callv = 0;
    if (cw == 0) {
        unsigned* cnt = (unsigned*)(myws + WS_CNT_MAIN) + b;
        callv = __hip_atomic_load(cnt, __ATOMIC_RELAXED, __HIP_MEMORY_SCOPE_SYSTEM);
        if (lane == 0) *(__attribute__((address_space(3))) unsigned*)(lds + CTRL_OFF + 8) = 0u;
    }
    char* obase = p.out[dest] + ((size_t)(rank * MS + row0) * NBLK + cg * 128) * 4;
    if (mt == 1) run_consumer<1>(p, lds, obase, cw, lane, cw == 0, callv, false, false);
    else run_consumer<2>(p, lds, obase, cw, lane, cw == 0, callv, second, false);
    if (cw == 0) STAMP(1);
    unsigned call = *(__attribute__((address_space(3))) unsigned*)(lds + CTRL_OFF);
    call = __builtin_amdgcn_readfirstlane(call);
    if (!local) {
        WAIT_VM(0);
        if (cw == 0) STAMP(2);
        if (lane == 0) {
            unsigned n = __hip_atomic_fetch_add((__attribute__((address_space(3))) unsigned*)(lds + CTRL_OFF + 8), 1u,
                                                __ATOMIC_RELAXED, __HIP_MEMORY_SCOPE_WORKGROUP);
            if (n == 3)
                __hip_atomic_store((unsigned*)(p.ws[dest] + WS_OFLAG) + rank * 32 + li, call, __ATOMIC_RELAXED,
                                   __HIP_MEMORY_SCOPE_SYSTEM);
        }
        return;
    }
    if (NX > 0 && second) {
        bool poison = *(__attribute__((address_space(3))) unsigned*)(lds + CTRL_OFF + 4) != 0u;
        char* o2 = p.out[rank] + ((size_t)(src2 * MS + MS - NX) * NBLK + cg * 128) * 4;
        run_consumer<(NX > 0 ? NX / 32 : 1)>(p, lds, o2, cw, lane, false, 0u, false, poison);
        if (cw == 0) STAMP(3);
    }
    if (li == 31 && cw == 0) {
        bool ok = wait_flags(p, WS_OFLAG, 0, true, call, lane);
        if (!ok) {
            WAIT_VM(0);
            if (lane == 0)
                __hip_atomic_store((float*)obase, __builtin_nanf(""), __ATOMIC_RELAXED, __HIP_MEMORY_SCOPE_SYSTEM);
        }
        STAMP(4);
    }
}

}

extern "C" void kernel_launch(int rank, int nranks, ncclComm_t comm, hipStream_t stream, void* const* d_in,
                              const int* in_sizes, int n_in, void* d_out, int out_size, const int* in_dtypes,
                              int out_dtype, void* const* d_out_all, void* const* d_ws_all, size_t ws_bytes) {
    (void)comm; (void)in_dtypes; (void)out_dtype; (void)d_out;
    if (nranks != NR || n_in != 4 || (unsigned)rank >= (unsigned)NR || in_sizes[0] != MS * KD ||
        in_sizes[1] != KD * ND || out_size != NR * MS * NBLK || ws_bytes < WS_NEED) {
        fprintf(stderr, "kernel_launch: unexpected problem shape\n");
        return;
    }
    Params p;
    p.x = (const char*)d_in[0];
    p.w = (const char*)d_in[1];
    p.sx = (const float*)d_in[2];
    p.sw = (const float*)d_in[3];
    for (int i = 0; i < NR; ++i) {
        p.out[i] = (char*)d_out_all[i];
        p.ws[i] = (char*)d_ws_all[i];
    }
    p.rank = rank;
    hipLaunchKernelGGL(fused_gemm_a2a, dim3(NWG), dim3(NTHREADS), 0, stream, p);
    HIP_CHECK(hipGetLastError());
}
```

```cpp
#include <hip/hip_runtime.h>
#include <rccl/rccl.h>
#include <cstdio>
#include <cstdint>

#define HIP_CHECK(expr) do { hipError_t _e = (expr); if (_e != hipSuccess) { \
    fprintf(stderr, "HIP error %s at %s:%d\n", hipGetErrorString(_e), __FILE__, __LINE__); \
    return; } } while (0)

#define NX_ROWS 88
#define NPUSH 4
#define XWIN 5
#define WIN 6
#define WTAIL 8
#define RAMP0 4
#define TD_LATE 500
#define PF_S0 3
#define PF_DELAY 60

namespace {

constexpr int NR = 8;
constexpr int MS = 512;
constexpr int KD = 4096;
constexpr int ND = 4096;
constexpr int NBLK = 512;
constexpr int NX = NX_ROWS;
constexpr int NP = NPUSH;
constexpr int RG_LATE = 4;
constexpr int BK = 128;
constexpr int NST = KD / BK;
constexpr int WB = BK * 128;
template <int MT> struct Geo {
    static constexpr int NL = 4 + MT;
    static constexpr int D = MT == 1 ? 7 : MT == 2 ? 6 : 5;
    static constexpr int SLOT = WB + MT * 32 * BK;
    static constexpr int LAG = MT == 1 ? 1 : 0;
    static constexpr int DE = D - LAG;
    static constexpr int NRD = 8 + 2 * MT;
};
constexpr int CTRL_OFF = 147456;
constexpr int PF_OFF = CTRL_OFF + 64;
constexpr int LDS_BYTES = PF_OFF + 8192;
static_assert(Geo<1>::D * Geo<1>::SLOT <= CTRL_OFF && Geo<2>::D * Geo<2>::SLOT <= CTRL_OFF &&
              Geo<3>::D * Geo<3>::SLOT <= CTRL_OFF, "ring");
constexpr int NWG = 256;
constexpr int NTHREADS = 8 * 64;

constexpr size_t WS_TMO = 2048;
constexpr size_t WS_OFLAG = 4096;
constexpr size_t WS_XFLAG = 8192;
constexpr size_t WS_CNT = 16384;
constexpr size_t WS_XSTAGE = 65536;
constexpr size_t XSTAGE_SRC = (size_t)96 * KD;
constexpr size_t WS_NEED = WS_XSTAGE + 2 * NR * XSTAGE_SRC;
constexpr unsigned long long DEADLINE = 50000000ull;

static_assert(NX % 8 == 0 && NX <= 96, "NX");
static_assert(NP >= 1 && NP <= 4 && (NX * 4) % (NP * 8) == 0, "NP");
constexpr int RROWS = MS - NX;
constexpr int LATE0 = 192;
static_assert(NP == 4 && LATE0 + 4 * 64 >= RROWS && LATE0 + 3 * 64 < RROWS, "the last tile is partly unused");
constexpr int MT2 = NX > 0 ? (NX + 31) / 32 : 1;
constexpr int XNI = NX > 0 ? NX * 4 / (NP * 8) : 1;

typedef int v2i __attribute__((ext_vector_type(2)));
typedef int v4i __attribute__((ext_vector_type(4)));
typedef int v8i __attribute__((ext_vector_type(8)));
typedef unsigned v4u __attribute__((ext_vector_type(4)));
typedef float v16f __attribute__((ext_vector_type(16)));
typedef __attribute__((address_space(3))) char* ldsp;
typedef volatile __attribute__((address_space(3))) unsigned* ldsw;

struct Params {
    const char* x;
    const char* w;
    int rank;
    int pad;
    char* myws;
    const float* sx;
    const float* sw;
    char* out[NR];
    char* ws[NR];
};

__device__ __forceinline__ unsigned long long rtc() { return __builtin_amdgcn_s_memrealtime(); }
#define STAMP(idx) do {} while (0)

#define WAIT_VM(n) asm volatile("s_waitcnt vmcnt(%0)" :: "i"(n) : "memory")
#define WAIT_LGKM0() asm volatile("s_waitcnt lgkmcnt(0)" ::: "memory")
#define WAIT_LGKM(n) asm volatile("s_waitcnt lgkmcnt(%0)" :: "i"(n) : "memory")
#define BAR() __builtin_amdgcn_s_barrier()

__device__ __forceinline__ __amdgpu_buffer_rsrc_t mkrsrc(const void* base, unsigned bytes) {
    return __builtin_amdgcn_make_buffer_rsrc(const_cast<void*>(base), 0, (int)bytes, 0x00020000);
}
__device__ __forceinline__ unsigned lds_word(ldsp lds, int i) {
    return __builtin_amdgcn_readfirstlane(*(ldsw)(lds + CTRL_OFF + 4 * i));
}

__device__ __forceinline__ bool flags_ok(v4u f, unsigned mask, unsigned call) {
    unsigned got = (f.x >= call ? 1u : 0u) | (f.y >= call ? 2u : 0u) | (f.z >= call ? 4u : 0u) | (f.w >= call ? 8u : 0u);
    return __all((got & mask) == mask);
}
__device__ __forceinline__ bool wait_flags(const Params& p, size_t region, unsigned mask, unsigned call, int lane) {
    unsigned* tmo = (unsigned*)(p.myws + WS_TMO);
    const char* fp = p.myws + region + lane * 16;
    unsigned long long t0 = rtc();
    for (;;) {
#pragma unroll 1
        for (int it = 0; it < 16; ++it) {
            v4u f;
            asm volatile("global_load_dwordx4 %0, %1, off sc0 sc1\n\ts_waitcnt vmcnt(0)" : "=&v"(f) : "v"(fp) : "memory");
            if (flags_ok(f, mask, call)) return true;
        }
        if (__hip_atomic_load(tmo, __ATOMIC_RELAXED, __HIP_MEMORY_SCOPE_AGENT) == call || rtc() - t0 > DEADLINE) {
            if (lane == 0) __hip_atomic_store(tmo, call, __ATOMIC_RELAXED, __HIP_MEMORY_SCOPE_AGENT);
            return false;
        }
    }
}

template <int AUX>
__device__ __forceinline__ void dma16(__amdgpu_buffer_rsrc_t r, ldsp dst, unsigned voff, unsigned soff) {
    __builtin_amdgcn_raw_ptr_buffer_load_lds(r, (__attribute__((address_space(3))) void*)dst, 16, voff, soff, 0, AUX);
}

template <int T, int AAUX>
__device__ __forceinline__ void load_stage(__amdgpu_buffer_rsrc_t rw, __amdgpu_buffer_rsrc_t ra, ldsp lds, int slot,
                                           int ks, int lw, unsigned vW, unsigned vA) {
    typedef Geo<T> G;
    ldsp sw = lds + slot * G::SLOT + lw * 4096;
    unsigned so = (unsigned)ks * (BK * ND) + lw * (32 * ND);
#pragma unroll
    for (int j = 0; j < 4; ++j) dma16<0>(rw, sw + j * 1024, vW, so + j * (8 * ND));
    ldsp sa = lds + slot * G::SLOT + WB + lw * 1024;
    unsigned sao = (unsigned)ks * BK + lw * (8 * KD);
#pragma unroll
    for (int m = 0; m < T; ++m) dma16<AAUX>(ra, sa + m * 4096, vA, sao + m * (32 * KD));
}

template <int N, int NL>
__device__ __forceinline__ void loader_tail() {
    WAIT_VM(N * NL);
    BAR();
    if constexpr (N > 0) loader_tail<N - 1, NL>();
}

template <int T, int AAUX, int S, int P0, int NRAMP>
__device__ __forceinline__ void loader_ramp(__amdgpu_buffer_rsrc_t rw, __amdgpu_buffer_rsrc_t ra, ldsp lds, int lw,
                                            unsigned vW, unsigned vA) {
    if constexpr (S < NRAMP) {
        constexpr int D = Geo<T>::D, I = P0 + 2 * S;
        WAIT_VM((I - S - 1) * Geo<T>::NL);
        BAR();
        load_stage<T, AAUX>(rw, ra, lds, I % D, I, lw, vW, vA);
        load_stage<T, AAUX>(rw, ra, lds, (I + 1) % D, I + 1, lw, vW, vA);
        static_assert(I + 1 <= S + Geo<T>::DE - 1, "slot still in use");
        loader_ramp<T, AAUX, S + 1, P0, NRAMP>(rw, ra, lds, lw, vW, vA);
    }
}

template <int T, int AAUX>
__device__ __forceinline__ void loader_tile(const char* wbase, const char* abase, ldsp lds, int lw, int lane) {
    typedef Geo<T> G;
    constexpr int NL = G::NL, D = G::D, DE = G::DE;
    __amdgpu_buffer_rsrc_t rw = mkrsrc(wbase, (unsigned)((KD - 1) * ND + 128));
    __amdgpu_buffer_rsrc_t ra = mkrsrc(abase, (unsigned)(T * 32 * KD));
    unsigned vW = (lane >> 3) * ND + (((lane & 7) ^ (((lane >> 4) & 3) << 1)) << 4);
    unsigned vA = (lane >> 3) * KD + (((lane & 7) ^ (((lw & 1) << 2) | (lane >> 4))) << 4);
    constexpr int P0 = RAMP0 < DE - 1 ? RAMP0 : DE - 1, NRAMP = DE - 1 - P0;
#pragma unroll
    for (int s = 0; s < P0; ++s) load_stage<T, AAUX>(rw, ra, lds, s, s, lw, vW, vA);
    loader_ramp<T, AAUX, 0, P0, NRAMP>(rw, ra, lds, lw, vW, vA);
    int slot = (NRAMP + DE - 1) % D;
#pragma unroll 1
    for (int s = NRAMP + DE - 1; s < NST; ++s) {
        WAIT_VM((DE - 2) * NL);
        BAR();
        load_stage<T, AAUX>(rw, ra, lds, slot, s, lw, vW, vA);
        slot = slot == D - 1 ? 0 : slot + 1;
    }
    loader_tail<DE - 2, NL>();
}

template <int MT>
struct Frags {
    v8i a[MT];
    v8i b[2];
};

template <int T>
__device__ __forceinline__ void read_frags(Frags<T>& f, ldsp sbase, unsigned bOff, unsigned aOff) {
    typedef __attribute__((address_space(3))) v2i* p2;
    typedef __attribute__((address_space(3))) v4i* p4;
#pragma unroll
    for (int n = 0; n < 2; ++n) {
        const unsigned bo = bOff ^ (unsigned)(32 * n);
        v2i t0 = __builtin_amdgcn_ds_read_tr8_b64_v2i32((p2)(sbase + bo + 0 * 128));
        v2i t1 = __builtin_amdgcn_ds_read_tr8_b64_v2i32((p2)(sbase + bo + 8 * 128));
        v2i t2 = __builtin_amdgcn_ds_read_tr8_b64_v2i32((p2)(sbase + bo + 16 * 128));
        v2i t3 = __builtin_amdgcn_ds_read_tr8_b64_v2i32((p2)(sbase + bo + 24 * 128));
        f.b[n] = v8i{t0.x, t0.y, t1.x, t1.y, t2.x, t2.y, t3.x, t3.y};
        if (n == 0) {
#pragma unroll
            for (int m = 0; m < T; ++m) {
                v4i lo = *(p4)(sbase + WB + m * 4096 + aOff);
                v4i hi = *(p4)(sbase + WB + m * 4096 + (aOff ^ 16u));
                f.a[m] = v8i{lo.x, lo.y, lo.z, lo.w, hi.x, hi.y, hi.z, hi.w};
            }
        }
    }
}

template <int MT>
__device__ __forceinline__ void mma(v16f (&acc)[MT][2], const Frags<MT>& f) {
#pragma unroll
    for (int n = 0; n < 2; ++n)
#pragma unroll
        for (int m = 0; m < MT; ++m)
            acc[m][n] = __builtin_amdgcn_mfma_scale_f32_32x32x64_f8f6f4(f.a[m], f.b[n], acc[m][n], 0, 0, 0, 0, 0, 0);
}

__device__ __forceinline__ void signal_tile(ldsp lds, unsigned* flag, int lane) {
    WAIT_VM(0);
    const unsigned call = lds_word(lds, 0);
    if (lane == 0) {
        unsigned n = __hip_atomic_fetch_add((__attribute__((address_space(3))) unsigned*)(lds + CTRL_OFF + 8), 1u,
                                            __ATOMIC_RELAXED, __HIP_MEMORY_SCOPE_WORKGROUP);
        if (n == 3) __hip_atomic_store(flag, call, __ATOMIC_RELAXED, __HIP_MEMORY_SCOPE_SYSTEM);
    }
}

template <int T>
__device__ __forceinline__ void consumer_tile(const Params& p, ldsp lds, v16f (&res)[T], int cw, int lane,
                                              bool first, unsigned prev, int src2) {
    typedef Geo<T> G;
    constexpr int SLOT = G::SLOT, D = G::D, MT = T;
    const int kh = cw & 1;
    int g = lane >> 4, q = (lane & 15) >> 1, pp = lane & 1;
    unsigned bOff = (64 * kh + 32 * (g >> 1) + q) * 128 + (((2 * cw + (g & 1)) ^ ((q >> 1) << 1)) << 4) + 8 * pp;
    int r = lane & 31, h = lane >> 5;
    unsigned aOff = r * 128 + (((4 * kh + 2 * h) ^ ((r >> 1) & 7)) << 4);
    v16f acc[MT][2];
#pragma unroll
    for (int m = 0; m < MT; ++m)
#pragma unroll
        for (int i = 0; i < 16; ++i) { acc[m][0][i] = 0.f; acc[m][1][i] = 0.f; }
    Frags<MT> f0, f1;
    v4u xf = {0u, 0u, 0u, 0u};
    BAR();
    if (first && cw == 0) STAMP(5);
    read_frags<T>(f0, lds, bOff, aOff);
    constexpr int LW = G::LAG ? G::NRD : 0;
    WAIT_LGKM(LW);
    __builtin_amdgcn_sched_barrier(0);
    int slot = 1;
#pragma unroll 1
    for (int s = 1; s < NST - 1; s += 2) {
        BAR();
        if (G::LAG) __builtin_amdgcn_sched_barrier(0);
        read_frags<T>(f1, lds + slot * SLOT, bOff, aOff);
        if (G::LAG) __builtin_amdgcn_sched_barrier(0);
        mma<MT>(acc, f0);
        if (first && cw == 0 && s == 15) {
            asm volatile("s_waitcnt vmcnt(0)" : "+v"(prev) :: "memory");
            if (lane == 0) {
                *(ldsw)(lds + CTRL_OFF) = prev + 1;
                __hip_atomic_store((unsigned*)(p.myws + WS_CNT) + blockIdx.x * 16, prev + 1, __ATOMIC_RELAXED,
                                   __HIP_MEMORY_SCOPE_SYSTEM);
            }
            WAIT_LGKM0();
        }
        if (T == 2 && src2 >= 0 && cw == 1) {
            if (s == 25) {
                const char* fp = p.myws + WS_XFLAG + lane * 16;
                asm volatile("global_load_dwordx4 %0, %1, off sc0 sc1" : "=v"(xf) : "v"(fp) : "memory");
            }
            if (s == 29) {
                asm volatile("s_waitcnt vmcnt(0)" : "+v"(xf) :: "memory");
                const bool ok = flags_ok(xf, lane == src2 * 8 ? (1u << NP) - 1u : 0u, lds_word(lds, 0));
                if (lane == 0) *(ldsw)(lds + CTRL_OFF + 12) = ok ? 1u : 0u;
                WAIT_LGKM0();
            }
        }
        WAIT_LGKM(LW);
        __builtin_amdgcn_sched_barrier(0);
        slot = slot == D - 1 ? 0 : slot + 1;
        BAR();
        if (first && cw == 0 && s == 7) STAMP(6);
        if (G::LAG) __builtin_amdgcn_sched_barrier(0);
        read_frags<T>(f0, lds + slot * SLOT, bOff, aOff);
        if (G::LAG) __builtin_amdgcn_sched_barrier(0);
        mma<MT>(acc, f1);
        WAIT_LGKM(LW);
        __builtin_amdgcn_sched_barrier(0);
        slot = slot == D - 1 ? 0 : slot + 1;
    }
    BAR();
    if (G::LAG) __builtin_amdgcn_sched_barrier(0);
    read_frags<T>(f1, lds + slot * SLOT, bOff, aOff);
    if (G::LAG) __builtin_amdgcn_sched_barrier(0);
    mma<MT>(acc, f0);
    mma<MT>(acc, f1);
    typedef float v4f __attribute__((ext_vector_type(4)));
    typedef __attribute__((address_space(3))) v4f* p4f;
    constexpr int XOFF = ((NST - 1) % D + 1) * SLOT;
    static_assert(XOFF + 4 * MT * 4096 <= D * SLOT && (NST - 1) % D >= 1, "exchange area");
    ldsp mine = lds + XOFF + cw * (MT * 4096) + lane * 16, other = lds + XOFF + (cw ^ 1) * (MT * 4096) + lane * 16;
#pragma unroll
    for (int m = 0; m < MT; ++m) {
        const v16f give = acc[m][1];
#pragma unroll
        for (int i = 0; i < 4; ++i)
            *(p4f)(mine + m * 4096 + i * 1024) = v4f{give[4 * i], give[4 * i + 1], give[4 * i + 2], give[4 * i + 3]};
    }
    WAIT_LGKM0();
    BAR();
#pragma unroll
    for (int m = 0; m < MT; ++m) {
        const v16f keep = acc[m][0];
#pragma unroll
        for (int i = 0; i < 4; ++i) {
            const v4f t = *(p4f)(other + m * 4096 + i * 1024);
#pragma unroll
            for (int j = 0; j < 4; ++j) res[m][4 * i + j] = keep[4 * i + j] + t[j];
        }
    }
    if (src2 >= 0) {
        WAIT_LGKM0();
        __builtin_amdgcn_sched_barrier(0);
        BAR();
    }
}

template <int T>
__device__ __forceinline__ void store_tile(char* obase, const v16f (&acc)[T], float scale, int cw, int lane,
                                           int nrows, bool poison, bool windowed) {
    __amdgpu_buffer_rsrc_t ro = mkrsrc(obase, (unsigned)((T * 32 - 1) * (NBLK * 4) + 512));
    unsigned vo = (lane >> 5) * (4 * NBLK * 4) + (32 * cw + (lane & 31)) * 4;
#pragma unroll
    for (int m = 0; m < T; ++m)
#pragma unroll
        for (int i = 0; i < 16; ++i) {
            if (32 * m + 8 * (i >> 2) < nrows) {
                float v = poison ? __builtin_nanf("") : acc[m][i] * scale;
                __builtin_amdgcn_raw_buffer_store_b32(__builtin_bit_cast(unsigned, v), ro, vo,
                                                      (32 * m + (i & 3) + 8 * (i >> 2)) * (NBLK * 4), 17);
                if (T == 2 && 16 * m + i < 32 - WTAIL && windowed) WAIT_VM(WIN - 1);
            }
        }
}

template <int T>
__device__ __forceinline__ void run_consumer(const Params& p, ldsp lds, char* obase, int cw, int lane, bool first,
                                             unsigned prev, int src2, float scale, int nrows,
                                             bool windowed) {
    v16f acc[T];
    consumer_tile<T>(p, lds, acc, cw, lane, first, prev, src2);
    const bool poison = !first && lds_word(lds, 1) != 0u;
    if (first && cw == 0) STAMP(7);
    store_tile<T>(obase, acc, scale, cw, lane, nrows, poison, windowed);
}

__global__ void __launch_bounds__(NTHREADS) fused_gemm_a2a(Params p) {
    __shared__ __attribute__((aligned(16))) char smem[LDS_BYTES];
    ldsp lds = (ldsp)smem;
    const int lane = threadIdx.x & 63;
    const int wave = __builtin_amdgcn_readfirstlane(threadIdx.x >> 6);
    const int b = blockIdx.x;
    const int sl = b & 7, li = b >> 3, cg = li & 3, rg = li >> 2;
    const int rank = p.rank;
    asm volatile("" :: "s"(rank), "s"(p.x), "s"(p.w), "s"(p.myws), "s"(p.ws[sl]), "s"(p.out[sl]), "s"(p.sx));
    const int dest = (rank + 1 + sl) & 7;
    const bool local = sl == 7;

    const bool pusher = !local && rg == 0;
    const int mt = !local && rg < 2 ? 1 : 2;
    const int row0 = local ? rg * 64 : rg < 2 ? rg * 32 : rg * 64 - 64;
    const int nrows = local ? 64 : min(64, RROWS - row0);
    const bool second = local && rg < NR - 1;
    const int src2 = (rank + 1 + rg) & 7;

    unsigned prev = 0;
    if (wave == 0 || pusher) {
        const char* cnt = p.myws + WS_CNT + b * 64;
        asm volatile("global_load_dword %0, %1, off sc0 sc1" : "=v"(prev) : "v"(cnt) : "memory");
    }
    if (pusher) {
        const unsigned off = (li * 8 + wave) * 1024;
        __amdgpu_buffer_rsrc_t rs = mkrsrc(p.x + (size_t)(MS - NX) * KD, (unsigned)(NX * KD));
        v4u v[XNI];
#pragma unroll
        for (int j = 0; j < XNI; ++j)
            v[j] = __builtin_amdgcn_raw_buffer_load_b128(rs, lane * 16, off + j * (NP * 8192), 0);
        asm volatile("s_waitcnt vmcnt(%1)" : "+v"(prev) : "i"(XNI) : "memory");
        const unsigned call = __builtin_amdgcn_readfirstlane(prev) + 1;
        char* dws = p.ws[sl];
        __amdgpu_buffer_rsrc_t rd =
            mkrsrc(dws + WS_XSTAGE + ((size_t)(call & 1) * NR + rank) * XSTAGE_SRC, (unsigned)XSTAGE_SRC);
#pragma unroll
        for (int j = 0; j < XNI; ++j) {
            __builtin_amdgcn_raw_buffer_store_b128(v[j], rd, lane * 16, off + j * (NP * 8192), 17);
            if (j == 0 && wave == 0) STAMP(15);
            if (j >= XWIN - 1 && j + 1 < XNI) WAIT_VM(XWIN - 1);
        }
        if (wave == 0) STAMP(8);
        WAIT_VM(0);
        BAR();
        if (wave == 0) {
            if (lane == 0)
                __hip_atomic_store((unsigned*)(dws + WS_XFLAG) + rank * 32 + li, call, __ATOMIC_RELAXED,
                                   __HIP_MEMORY_SCOPE_SYSTEM);
            STAMP(9);
        }
    }

    if (local ? rg == 7 : rg >= RG_LATE) {
        const unsigned long long t0 = rtc();
        if (wave < 4 || local) {
            const int u = local ? (li - 28) * 8 + wave : (li - 16) * 4 + wave;
            __amdgpu_buffer_rsrc_t rw = mkrsrc(p.w + dest * NBLK, (unsigned)((KD - 1) * ND + NBLK));
            __amdgpu_buffer_rsrc_t rx = local ? rw : mkrsrc(p.x, (unsigned)(MS * KD));
            const unsigned vw = (2 * u + (lane >> 5)) * ND + (lane & 31) * 16;
            const unsigned vx = local ? vw + 64 * ND : (8 * u + (lane >> 3)) * KD + (lane & 7) * 16;
            const unsigned xstep = local ? BK * ND : BK;
            while (rtc() - t0 < PF_DELAY) __builtin_amdgcn_s_sleep(2);
            const unsigned m0v = PF_OFF + wave * 1024;
            __amdgpu_buffer_rsrc_t rl = mkrsrc(p.w + rank * NBLK, (unsigned)((KD - 1) * ND + NBLK));
            int turn = local ? -1000 : sl;
#define PF_LOAD(v, r, so)                                                                                  \
    asm volatile("s_mov_b32 %0, m0\n\ts_mov_b32 m0, %1\n\ts_nop 0\n\t"                                       \
                 "buffer_load_dwordx4 %2, %3, %4 offen lds\n\ts_mov_b32 m0, %0"                              \
                 : "=&s"(keep) : "s"(m0v), "v"(v), "s"(r), "s"(so) : "memory")
            unsigned keep;
            unsigned sx2 = local ? PF_S0 * (BK * ND) : PF_S0 * BK;
#pragma unroll 1
            for (unsigned so = PF_S0 * (BK * ND); so < NST * (BK * ND); so += BK * ND) {
                asm volatile("" : "+s"(so), "+s"(sx2));
                PF_LOAD(vw, rw, so);
                PF_LOAD(vx, rx, sx2);
                if (turn == 0) {
                    PF_LOAD(vw, rl, so);
                    turn = NR - 1;
                }
                --turn;
                sx2 += xstep;
            }
            if (wave >= 4) WAIT_VM(0);
            if (wave == 0) STAMP(8);
        } else {
            while (rtc() - t0 < TD_LATE) __builtin_amdgcn_s_sleep(8);
        }
    }

    if (wave >= 4) {
        const int lw = wave - 4;
        const char* wbase = p.w + dest * NBLK + cg * 128;
        const char* abase = p.x + (size_t)row0 * KD;
        if (lw == 0) STAMP(14);
        if (mt == 1) loader_tile<1, 0>(wbase, abase, lds, lw, lane);
        else loader_tile<2, 0>(wbase, abase, lds, lw, lane);
        if (second) {
            const unsigned call = lds_word(lds, 0);
            BAR();
            BAR();
            bool ok = lds_word(lds, 3) != 0u;
            if (!ok) ok = wait_flags(p, WS_XFLAG, lane == src2 * 8 ? (1u << NP) - 1u : 0u, call, lane);
            if (!ok && lane == 0) *(ldsw)(lds + CTRL_OFF + 4) = 1u;
            asm volatile("s_waitcnt vmcnt(0) lgkmcnt(0)" ::: "memory");
            const char* a2 = p.myws + WS_XSTAGE + ((size_t)(call & 1) * NR + src2) * XSTAGE_SRC;
            loader_tile<MT2, 17>(wbase, a2, lds, lw, lane);
        }
        return;
    }

    const int cw = wave;
    typedef const __attribute__((address_space(4))) float* cfp;
    float sx = *(cfp)p.sx, sw = *(cfp)p.sw;
    if (cw == 0 && lane == 0) {
        *(ldsw)(lds + CTRL_OFF + 4) = 0u;
        *(ldsw)(lds + CTRL_OFF + 8) = 0u;
    }
    char* obase = p.out[sl] + ((size_t)row0 * NBLK + cg * 128) * 4;
    asm volatile("" : "+s"(sx), "+s"(sw));
    const float scale = sx * sw;
    if (mt == 1) run_consumer<1>(p, lds, obase, cw, lane, true, prev, -1, scale, 32, false);
    else run_consumer<2>(p, lds, obase, cw, lane, true, prev, second ? src2 : -1, scale, nrows, !local);
    if (cw == 0) STAMP(1);
    if (!local) {
        signal_tile(lds, (unsigned*)(p.ws[sl] + WS_OFLAG) + rank * 32 + li, lane);
        if (cw == 0) STAMP(2);
        return;
    }
    if (second) {
        char* o2 = p.out[7] + ((ptrdiff_t)((src2 - rank) * MS + MS - NX) * NBLK + cg * 128) * 4;
        run_consumer<MT2>(p, lds, o2, cw, lane, false, 0u, -1, scale, NX, false);
        if (cw == 0) STAMP(3);
    }
    const unsigned call = lds_word(lds, 0);
    if (li == 31 && cw == 0) {
        bool ok = wait_flags(p, WS_OFLAG, (lane >> 3) != rank ? 0xFu : 0u, call, lane);
        if (!ok) {
            WAIT_VM(0);
            if (lane == 0)
                __hip_atomic_store((float*)obase, __builtin_nanf(""), __ATOMIC_RELAXED, __HIP_MEMORY_SCOPE_SYSTEM);
        }
        STAMP(4);
    }
}

}

extern "C" void kernel_launch(int rank, int nranks, ncclComm_t comm, hipStream_t stream, void* const* d_in,
                              const int* in_sizes, int n_in, void* d_out, int out_size, const int* in_dtypes,
                              int out_dtype, void* const* d_out_all, void* const* d_ws_all, size_t ws_bytes) {
    (void)comm; (void)in_dtypes; (void)out_dtype; (void)d_out;
    if (nranks != NR || n_in != 4 || (unsigned)rank >= (unsigned)NR || in_sizes[0] != MS * KD ||
        in_sizes[1] != KD * ND || out_size != NR * MS * NBLK || ws_bytes < WS_NEED) {
        fprintf(stderr, "kernel_launch: unexpected problem shape\n");
        return;
    }
    Params p;
    p.x = (const char*)d_in[0];
    p.w = (const char*)d_in[1];
    p.sx = (const float*)d_in[2];
    p.sw = (const float*)d_in[3];
    for (int i = 0; i < NR; ++i) {
        int r = (rank + 1 + i) % NR;
        p.out[i] = (char*)d_out_all[r] + (size_t)rank * MS * NBLK * 4;
        p.ws[i] = (char*)d_ws_all[r];
    }
    p.myws = (char*)d_ws_all[rank];
    p.rank = rank;
    p.pad = 0;
    hipLaunchKernelGGL(fused_gemm_a2a, dim3(NWG), dim3(NTHREADS), 0, stream, p);
    HIP_CHECK(hipGetLastError());
}
```
